```python
import numpy as np

kernel_name = "gpu_a2a_gemm_m4096_k4096_n4096_f32_gelu"


ABSMAX_RTOL = 0.001

COMPUTE_DTYPE = "float32"
EPILOGUE = "gelu"
IN_QUANT = None
OUT_QUANT = None
IN_DTYPES = {"x": "float32", "w_mat": "float32"}
WORLD_SIZE = 8

def setup_inputs():

    x = np.random.randn(4096, 4096).astype(np.float32)
    w_mat = (np.random.randn(4096, 4096) / np.sqrt(4096)).astype(np.float32)
    return {"x": x, "w_mat": w_mat}

def _epilogue(y):
    c = 0.7978845608028654
    return (0.5 * y * (1.0 + np.tanh(c * (y + 0.044715 * y ** 3)))).astype(np.float32)

def reference(x, w_mat):
    y = (x @ w_mat).astype(np.float32)
    return _epilogue(y)
```

```cpp
#include <hip/hip_runtime.h>
#include <rccl/rccl.h>
#include <cstdio>

typedef _Float16 half8 __attribute__((ext_vector_type(8)));
typedef float f32x4 __attribute__((ext_vector_type(4)));
typedef unsigned u32x16 __attribute__((ext_vector_type(16)));
typedef __attribute__((address_space(3))) int lds_int;
typedef __attribute__((address_space(3))) half8 lds_h8;

namespace {
constexpr int NR = 8;
constexpr int ND = 4096;
constexpr int MS = 512;
constexpr size_t WS_TMO   = 0;
constexpr size_t WS_CNT   = 4096;
constexpr size_t WS_FLAG  = 64 * 1024;
constexpr size_t WS_PFLAG = 128 * 1024;
constexpr size_t WS_XCC   = 192 * 1024;
constexpr size_t WS_A     = 1 << 20;
constexpr size_t A_BYTES  = (size_t)4 << 20;
constexpr size_t WS_NEED  = WS_A + 2 * A_BYTES;
constexpr size_t WS_TS    = (size_t)40 << 20;
constexpr int SYNC_OFF = 32 * 4096;
constexpr int LDS_BYTES = SYNC_OFF + 1024;
constexpr unsigned long long DEADLINE_TICKS = 100000000ull;
constexpr int NLOAD = 2;
#define WDEPTH 2
#define HOLD_LEAD 100
#define SLOT_T0 150
#define SLOT_DT 28

__device__ __forceinline__ float gelu_tanh(float y) {
  float u = 0.7978845608028654f * (y + 0.044715f * y * y * y);
  float e = __expf(2.f * u);
  float t = 1.f - 2.f / (e + 1.f);
  return 0.5f * y * (1.f + t);
}
}

#define TS(k)
#define TS_DECL
#define TS_OUT

#define LDWR(dst, j, ptr) asm volatile("global_load_dwordx4 %0, %1, %2" : "+v"(dst) : "v"(woff[j]), "s"(ptr))
#define LDAR(dst, ptr) asm volatile("global_load_dwordx4 %0, %1, %2 sc0 sc1" : "+v"(dst) : "v"(aoff), "s"(ptr))
#define REGS8(r) "+v"(r[0]), "+v"(r[1]), "+v"(r[2]), "+v"(r[3]), "+v"(r[4]), "+v"(r[5]), "+v"(r[6]), "+v"(r[7])
#define H8(x) __builtin_bit_cast(half8, x)
#define LDS_LD(i) __hip_atomic_load(&sync[i], __ATOMIC_RELAXED, __HIP_MEMORY_SCOPE_WORKGROUP)
#define LDS_ST(i, v) __hip_atomic_store(&sync[i], (int)(v), __ATOMIC_RELAXED, __HIP_MEMORY_SCOPE_WORKGROUP)

__global__ __launch_bounds__(768, 3) void a2a_gemm_gelu_kernel(const float* __restrict__ X, const float* __restrict__ W,
                                                               float* __restrict__ C, void* const* __restrict__ ws_all,
                                                               int rank) {
  extern __shared__ __attribute__((aligned(16))) char smem[];
  TS_DECL; TS(0);
  const unsigned long long t_start = __builtin_amdgcn_s_memrealtime();
  const int wave = __builtin_amdgcn_readfirstlane(threadIdx.x >> 6), lane = threadIdx.x & 63;
  const int g = lane >> 4, c = lane & 15;
  const int bid = blockIdx.x;
  const int s = (bid >> 3) & 1, mh = (bid >> 4) & 1, copy = bid & 1, nt = (bid & 7) + 8 * (bid >> 5);
  const int pbid = bid ^ 8;
  char* ws = (char*)ws_all[rank];
  lds_int* sync = (lds_int*)(smem + SYNC_OFF);
  unsigned* tmo = (unsigned*)(ws + WS_TMO);
  if (threadIdx.x < 96) sync[threadIdx.x] = 0;

  auto gk = [&](int kq) -> int {
    if (kq < 8) return rank * 16 + s + 2 * kq;
    int p = kq - 8;
    int j = (p * 9363) >> 16;
    int q = p - j * 7;
    return ((rank + 1 + q) & 7) * 16 + s + 2 * j;
  };

  if (wave < 8) {
    const int n0 = nt * 64;
    const unsigned aoff = (unsigned)(wave * 1024 + lane * 16);
    f32x4 ra[8], rb[8];
#pragma unroll
    for (int j = 0; j < 8; j++) { ra[j] = f32x4{0, 0, 0, 0}; rb[j] = f32x4{0, 0, 0, 0}; }
    asm volatile("s_waitcnt lgkmcnt(0)" ::: "memory");
    __builtin_amdgcn_s_barrier();
    unsigned call = 0;
    auto waitsync = [&](int pneed, int ownneed, int peerneed) {
      const int need = lane == 0 ? ownneed : (lane == 1 ? peerneed + 1 : (lane < 4 ? (pneed + NLOAD + 1 - lane) / NLOAD : 0));
      while (true) {
        int v = LDS_LD(lane & 7);
        bool ok = lane > 3 || v >= need;
        bool ab = lane == 4 && v != 0;
        if (__any(ab) || __all(ok)) break;
        __builtin_amdgcn_s_sleep(1);
      }
      asm volatile("" ::: "memory");
    };
    constexpr int OWNNEED[2] = {4, 8};
    constexpr int PEERNEED[16] = {-1, -1, 0, 1, 1, 2, 2, 3, 3, 4, 5, 5, 6, 6, 7, 7};
    f32x4 accS[4], accK[4];
#pragma unroll
    for (int n = 0; n < 4; n++) { accS[n] = f32x4{0, 0, 0, 0}; accK[n] = f32x4{0, 0, 0, 0}; }
    while ((call = (unsigned)__builtin_amdgcn_readfirstlane(LDS_LD(80))) == 0u) __builtin_amdgcn_s_sleep(1);
    const char* abase = ws + WS_A + (size_t)(call & 1) * A_BYTES + (size_t)mh * 16384;
    auto item_ptr = [&](int i) { return abase + (size_t)gk(i >> 1) * 32768 + (size_t)((i ^ s ^ 1) & 1) * 8192; };
    TS(1);
    waitsync(0, OWNNEED[0], PEERNEED[0]);
#pragma unroll
    for (int k = 0; k < 8; k++) { const char* p = item_ptr(k); LDAR(ra[k], p); }
    waitsync(0, OWNNEED[1], PEERNEED[1]);
#pragma unroll
    for (int k = 0; k < 8; k++) { const char* p = item_ptr(8 + k); LDAR(rb[k], p); }
    TS(2);
#define CHUNK(cc, R)                                                                                          \
    {                                                                                                         \
      waitsync(4 * (cc) + 4, 0, (cc) < 14 ? PEERNEED[(cc) < 14 ? (cc) + 2 : 0] : -1); \
      TS(8 + 2 * (cc));                                                                                       \
      const lds_h8* sp = (const lds_h8*)(smem + ((cc) & 7) * 4 * 4096 + lane * 16);                           \
      half8 b[2][4];                                                                                          \
      _Pragma("unroll") for (int n = 0; n < 4; n++) b[0][n] = sp[n * 64];                                     \
      _Pragma("unroll") for (int kk = 0; kk < 4; kk++) {                                                      \
        if (kk < 3) { _Pragma("unroll") for (int n = 0; n < 4; n++) b[(kk + 1) & 1][n] = sp[(kk + 1) * 256 + n * 64]; } \
        asm volatile("s_waitcnt vmcnt(%1)" : "+v"(R[2 * kk]) : "i"((cc) < 14 ? 15 : ((cc) == 14 ? 15 - 2 * kk : 7 - 2 * kk))); \
        _Pragma("unroll") for (int n = 0; n < 4; n++)                                                         \
            accS[n] = __builtin_amdgcn_mfma_f32_16x16x32_f16(H8(R[2 * kk]), b[kk & 1][n], accS[n], 0, 0, 0);  \
        __builtin_amdgcn_sched_barrier(0);                                                                    \
        if ((cc) < 14) { const char* p = item_ptr(((cc) + 2) * 8 + 2 * kk); LDAR(R[2 * kk], p); }             \
        asm volatile("s_waitcnt vmcnt(%1)" : "+v"(R[2 * kk + 1]) : "i"((cc) < 14 ? 15 : ((cc) == 14 ? 14 - 2 * kk : 6 - 2 * kk))); \
        _Pragma("unroll") for (int n = 0; n < 4; n++)                                                         \
            accK[n] = __builtin_amdgcn_mfma_f32_16x16x32_f16(H8(R[2 * kk + 1]), b[kk & 1][n], accK[n], 0, 0, 0); \
        __builtin_amdgcn_sched_barrier(0);                                                                    \
        if ((cc) < 14) { const char* p = item_ptr(((cc) + 2) * 8 + 2 * kk + 1); LDAR(R[2 * kk + 1], p); }     \
      }                                                                                                       \
      if ((cc) < 8 && lane == 0) LDS_ST(72 + wave, (cc) + 1);                                                 \
      TS(9 + 2 * (cc));                                                                                       \
    }
    CHUNK(0, ra); CHUNK(1, rb); CHUNK(2, ra); CHUNK(3, rb); CHUNK(4, ra); CHUNK(5, rb); CHUNK(6, ra); CHUNK(7, rb);
    CHUNK(8, ra); CHUNK(9, rb); CHUNK(10, ra); CHUNK(11, rb); CHUNK(12, ra); CHUNK(13, rb); CHUNK(14, ra); CHUNK(15, rb);
    TS(3);
    int mode;
    while ((mode = __builtin_amdgcn_readfirstlane(LDS_LD(67))) == 0 && __builtin_amdgcn_readfirstlane(LDS_LD(4)) == 0)
      __builtin_amdgcn_s_sleep(1);
    float* crow = C + (size_t)(mh * 256 + wave * 16 + g * 4) * ND + n0 + c * 4;
    float* cS = crow + (size_t)((s ^ 1) * 128) * ND;
    float* cK = crow + (size_t)(s * 128) * ND;
#pragma unroll
    for (int r = 0; r < 4; r++) {
      f32x4 o = {accS[0][r], accS[1][r], accS[2][r], accS[3][r]};
      if (mode == 1) asm volatile("global_store_dwordx4 %0, %1, off\n\ts_nop 1" ::"v"(cS + (size_t)r * ND), "v"(o) : "memory");
      else asm volatile("global_store_dwordx4 %0, %1, off sc1\n\ts_nop 1" ::"v"(cS + (size_t)r * ND), "v"(o) : "memory");
    }
    asm volatile("s_waitcnt vmcnt(0)" ::: "memory");
    if (lane == 0) __hip_atomic_store((unsigned*)(ws + WS_PFLAG + (size_t)pbid * 128 + wave * 4), call, __ATOMIC_RELAXED, __HIP_MEMORY_SCOPE_AGENT);
    TS(4);
    bool bad = false;
    {
      const unsigned* pf = (const unsigned*)(ws + WS_PFLAG + (size_t)bid * 128 + wave * 4);
      for (unsigned it = 1; (int)(__hip_atomic_load(pf, __ATOMIC_RELAXED, __HIP_MEMORY_SCOPE_AGENT) - call) < 0; it++) {
        if ((it & 63) == 0 && (__hip_atomic_load(tmo, __ATOMIC_RELAXED, __HIP_MEMORY_SCOPE_AGENT) == call ||
                               __builtin_amdgcn_s_memrealtime() - t_start > DEADLINE_TICKS)) {
          __hip_atomic_store(tmo, call, __ATOMIC_RELAXED, __HIP_MEMORY_SCOPE_AGENT);
          bad = true;
          break;
        }
      }
    }
    TS(5);
#pragma unroll
    for (int r = 0; r < 4; r++) asm volatile("global_load_dwordx4 %0, %1, off sc1" : "+v"(ra[r]) : "v"(cK + (size_t)r * ND));
    asm volatile("s_waitcnt vmcnt(0)" : REGS8(ra));
    TS(6);
    bad = bad || LDS_LD(4) != 0;
#pragma unroll
    for (int r = 0; r < 4; r++) {
      f32x4 o;
#pragma unroll
      for (int n = 0; n < 4; n++) o[n] = bad ? __builtin_nanf("") : gelu_tanh(accK[n][r] + ra[r][n]);
      *(f32x4*)(cK + (size_t)r * ND) = o;
    }
  } else if (wave > 9) {
    const int ld = wave - 10;
    unsigned woff[8];
#pragma unroll
    for (int j = 0; j < 8; j++) woff[j] = (unsigned)((g * 4 + (j >> 2) * 16 + (j & 3)) * ND + nt * 64 + c * 4) * 4u;
    f32x4 wr[WDEPTH][8];
#pragma unroll
    for (int q = 0; q < WDEPTH; q++)
#pragma unroll
      for (int j = 0; j < 8; j++) wr[q][j] = f32x4{0, 0, 0, 0};
    asm volatile("s_waitcnt lgkmcnt(0)" ::: "memory");
    __builtin_amdgcn_s_barrier();
    auto issue = [&](f32x4 (&w)[8], int kq) {
      const char* p = (const char*)W + (size_t)gk(kq) * (32 * ND * 4);
#pragma unroll
      for (int j = 0; j < 8; j++) LDWR(w[j], j, p);
    };
    const int cidx = lane < 8 ? 72 + lane : (lane == 10 ? 4 : 81);
    constexpr int NK = 64 / NLOAD;
#pragma unroll
    for (int i = 0; i < WDEPTH; i++) issue(wr[i], ld + i * NLOAD);
#pragma unroll
    for (int i = 0; i < NK; i++) {
      const int kq = ld + i * NLOAD;
      const int left = NK - 1 - i < WDEPTH - 1 ? NK - 1 - i : WDEPTH - 1;
      int cv = LDS_LD(cidx);
      asm volatile("s_waitcnt vmcnt(%8)" : REGS8(wr[i % WDEPTH]) : "i"(left * 8));
      half8 b[4];
#pragma unroll
      for (int n = 0; n < 4; n++)
#pragma unroll
        for (int j = 0; j < 8; j++) b[n][j] = (_Float16)wr[i % WDEPTH][j][n];
      const int need = i >= 32 / NLOAD ? ((kq - 32) >> 2) + 1 : 0;
      while (!__all(lane < 8 ? cv >= need : (lane != 9 || cv == 0)) && !__any(lane == 10 && cv != 0)) {
        __builtin_amdgcn_s_sleep(1);
        cv = LDS_LD(cidx);
      }
      if (i + WDEPTH < NK) issue(wr[i % WDEPTH], kq + WDEPTH * NLOAD);
      lds_h8* d = (lds_h8*)(smem + (kq & 31) * 4096 + lane * 16);
#pragma unroll
      for (int n = 0; n < 4; n++) d[n * 64] = b[n];
      asm volatile("" ::: "memory");
      if (lane == 0) LDS_ST(2 + ld, i + 1);
    }
    TS(1);
  } else if (wave == 8) {
    const int d = bid & 7, n = bid >> 3;
    const int t = n >> 1, uh = n & 1;
    const float* xp = X + (size_t)(d * MS + uh * 256 + c) * MS + t * 32 + g * 4;
    f32x4 xv[32];
#pragma unroll
    for (int r = 0; r < 16; r++) {
      xv[r * 2] = *(const f32x4*)(xp + (size_t)r * 16 * MS);
      xv[r * 2 + 1] = *(const f32x4*)(xp + (size_t)r * 16 * MS + 16);
    }
    __builtin_amdgcn_sched_barrier(0);
    asm volatile("s_waitcnt lgkmcnt(0)" ::: "memory");
    __builtin_amdgcn_s_barrier();
    __builtin_amdgcn_sched_barrier(0);
    half8 h[16];
#pragma unroll
    for (int f = 0; f < 16; f++)
#pragma unroll
      for (int j = 0; j < 8; j++) h[f][j] = (_Float16)xv[f * 2 + (j >> 2)][j & 3];
    unsigned call;
    while ((call = (unsigned)__builtin_amdgcn_readfirstlane(LDS_LD(80))) == 0u) __builtin_amdgcn_s_sleep(1);
    TS(1);
    char* dws = (char*)ws_all[d];
    char* dst = dws + WS_A + (size_t)(call & 1) * A_BYTES + (size_t)(rank * 16 + t) * 32768 + uh * 16384 + lane * 16;
    unsigned* fl = (unsigned*)(dws + WS_FLAG + ((uh * 2 + (t & 1)) * 2 + (lane & 1)) * 256 + (rank * 8 + (t >> 1)) * 4);
    const unsigned long long slot = (d == rank) ? 0ull : (unsigned long long)(SLOT_T0 + n * SLOT_DT);
    {
    while (__builtin_amdgcn_s_memrealtime() - t_start + HOLD_LEAD < slot) __builtin_amdgcn_s_sleep(1);
    if (lane == 0) LDS_ST(81, 1);
    while (__builtin_amdgcn_s_memrealtime() - t_start < slot) __builtin_amdgcn_s_sleep(1);
    TS(2);
#pragma unroll
    for (int f = 0; f < 16; f++)
      asm volatile("global_store_dwordx4 %0, %1, off sc0 sc1\n\ts_nop 1" ::"v"(dst + f * 1024), "v"(h[f]) : "memory");
    if (lane == 0) LDS_ST(81, 0);
    asm volatile("s_waitcnt vmcnt(0)" ::: "memory");
    if (lane < 2) __hip_atomic_store(fl, call, __ATOMIC_RELAXED, __HIP_MEMORY_SCOPE_SYSTEM);
    TS(3);
    }
  } else {
    unsigned* cntp = (unsigned*)(ws + WS_CNT + (size_t)bid * 128);
    const unsigned call_v = __hip_atomic_load(cntp, __ATOMIC_RELAXED, __HIP_MEMORY_SCOPE_AGENT) + 1u;
    asm volatile("s_waitcnt lgkmcnt(0)" ::: "memory");
    __builtin_amdgcn_s_barrier();
    const unsigned call = __builtin_amdgcn_readfirstlane(call_v);
    if (lane == 0) LDS_ST(80, call);
    unsigned xcc;
    asm volatile("s_getreg_b32 %0, hwreg(HW_REG_XCC_ID, 0, 4)" : "=s"(xcc));
    const unsigned xtag = (call << 8) | xcc;
    if (lane == 0) __hip_atomic_store((unsigned*)(ws + WS_XCC + (size_t)bid * 128), xtag, __ATOMIC_RELAXED, __HIP_MEMORY_SCOPE_AGENT);
    const char* myfl = ws + WS_FLAG + ((mh * 2 + s) * 2 + copy) * 256;
    const unsigned* xp = (const unsigned*)(ws + WS_XCC + (size_t)pbid * 128);
    bool moded = false;
    for (unsigned it = 1;; it++) {
      unsigned long long okm;
      {
        unsigned v = __hip_atomic_load((const unsigned*)myfl + lane, __ATOMIC_RELAXED, __HIP_MEMORY_SCOPE_SYSTEM);
        okm = __ballot((int)(v - call) >= 0);
      }
      if (!moded) {
        unsigned pv = __builtin_amdgcn_readfirstlane(__hip_atomic_load(xp, __ATOMIC_RELAXED, __HIP_MEMORY_SCOPE_AGENT));
        if ((pv >> 8) == (call & 0xffffffu)) {
          if (lane == 0) LDS_ST(67, pv == xtag ? 1 : 2);
          moded = true;
        }
      }
      int own = 0, peer = 8;
#pragma unroll
      for (int p = 0; p < 8; p++) {
        const int cnt = __builtin_ctz(~(unsigned)((okm >> (8 * p)) & 0xff));
        if (p == rank) own = cnt; else peer = cnt < peer ? cnt : peer;
      }
      if (lane < 2) LDS_ST(lane, lane == 0 ? own : peer);
      if (moded && okm == ~0ull) break;
      if ((it & 63) == 0 && (__hip_atomic_load(tmo, __ATOMIC_RELAXED, __HIP_MEMORY_SCOPE_AGENT) == call ||
                             __builtin_amdgcn_s_memrealtime() - t_start > DEADLINE_TICKS)) {
        __hip_atomic_store(tmo, call, __ATOMIC_RELAXED, __HIP_MEMORY_SCOPE_AGENT);
        LDS_ST(4, 1);
        break;
      }
    }
    TS(6);
    if (lane == 0) __hip_atomic_store(cntp, call, __ATOMIC_RELAXED, __HIP_MEMORY_SCOPE_AGENT);
  }
  TS_OUT;
}

extern "C" void kernel_launch(int rank, int nranks, ncclComm_t comm, hipStream_t stream, void* const* d_in,
                              const int* in_sizes, int n_in, void* d_out, int out_size, const int* in_dtypes,
                              int out_dtype, void* const* d_out_all, void* const* d_ws_all, size_t ws_bytes) {
  (void)comm; (void)in_sizes; (void)n_in; (void)out_size; (void)in_dtypes; (void)out_dtype; (void)d_out_all;
  if (nranks != NR || (unsigned)rank >= (unsigned)NR || ws_bytes < WS_NEED) {
    fprintf(stderr, "kernel_launch: unsupported configuration\n");
    return;
  }
  static bool init = false;
  if (!init) {
    hipError_t e = hipFuncSetAttribute((const void*)a2a_gemm_gelu_kernel, hipFuncAttributeMaxDynamicSharedMemorySize, LDS_BYTES);
    if (e != hipSuccess) { fprintf(stderr, "hipFuncSetAttribute: %s\n", hipGetErrorString(e)); return; }
    init = true;
  }
  a2a_gemm_gelu_kernel<<<256, 768, LDS_BYTES, stream>>>((const float*)d_in[0], (const float*)d_in[1], (float*)d_out,
                                                        d_ws_all, rank);
}
```

```cpp
#include <hip/hip_runtime.h>
#include <rccl/rccl.h>
#include <cstdio>

typedef _Float16 half8 __attribute__((ext_vector_type(8)));
typedef float f32x4 __attribute__((ext_vector_type(4)));
typedef __attribute__((address_space(3))) int lds_int;
typedef __attribute__((address_space(3))) half8 lds_h8;

namespace {
constexpr int NR = 8;
constexpr int ND = 4096;
constexpr int MS = 512;
constexpr size_t WS_TMO   = 0;
constexpr size_t WS_CNT   = 4096;
constexpr size_t WS_FLAG  = 64 * 1024;
constexpr size_t WS_PFLAG = 128 * 1024;
constexpr size_t WS_XCC   = 192 * 1024;
constexpr size_t WS_A     = 1 << 20;
constexpr size_t A_BYTES  = (size_t)4 << 20;
constexpr size_t WS_PART  = WS_A + 2 * A_BYTES;
constexpr size_t WS_NEED  = WS_PART + (size_t)256 * 3 * 32768;
constexpr size_t WS_TS    = (size_t)40 << 20;
constexpr int SYNC_OFF = 32 * 4096;
constexpr int LDS_BYTES = SYNC_OFF + 1024;
constexpr unsigned long long DEADLINE_TICKS = 100000000ull;
constexpr int NLOAD = 2;
#define WDEPTH 3
#define WSTART 0
#define WRAMP 0
#define HOLD_LEAD 100
#define SLOT_T0 150
#define SLOT_DT 28

__device__ __forceinline__ float gelu_tanh(float y) {
  float u = 0.7978845608028654f * (y + 0.044715f * y * y * y);
  float e = __expf(2.f * u);
  float t = 1.f - 2.f / (e + 1.f);
  return 0.5f * y * (1.f + t);
}
}

#define TS(k)
#define TS_DECL
#define TS_OUT

#define LDWR(dst, j, ptr) asm volatile("global_load_dwordx4 %0, %1, %2" : "+v"(dst) : "v"(woff[j]), "s"(ptr))
#define LDAR(dst, ptr) asm volatile("global_load_dwordx4 %0, %1, %2 sc0 sc1" : "+v"(dst) : "v"(aoff), "s"(ptr))
#define LDPR(dst, ptr, off) asm volatile("global_load_dwordx4 %0, %1, %2 offset:%3 sc1" : "+v"(dst) : "v"(poff), "s"(ptr), "i"(off))
#define REGS8(r) "+v"(r[0]), "+v"(r[1]), "+v"(r[2]), "+v"(r[3]), "+v"(r[4]), "+v"(r[5]), "+v"(r[6]), "+v"(r[7])
#define H8(x) __builtin_bit_cast(half8, x)
#define LDS_LD(i) __hip_atomic_load(&sync[i], __ATOMIC_RELAXED, __HIP_MEMORY_SCOPE_WORKGROUP)
#define LDS_ST(i, v) __hip_atomic_store(&sync[i], (int)(v), __ATOMIC_RELAXED, __HIP_MEMORY_SCOPE_WORKGROUP)

__global__ __launch_bounds__(768, 3) void a2a_gemm_gelu_kernel(const float* __restrict__ X, const float* __restrict__ W,
                                                               float* __restrict__ C, void* const* __restrict__ ws_all,
                                                               int rank) {
  extern __shared__ __attribute__((aligned(16))) char smem[];
  TS_DECL; TS(0);
  const unsigned long long t_start = __builtin_amdgcn_s_memrealtime();
  const int wave = __builtin_amdgcn_readfirstlane(threadIdx.x >> 6), lane = threadIdx.x & 63;
  const int g = lane >> 4, c = lane & 15;
  const int bid = blockIdx.x;
  const int s = (bid >> 3) & 3, copy = bid & 1, nt = (bid & 7) + 8 * (bid >> 5);
  auto partner = [&](int rel) { return (bid & ~24) | (((s + rel) & 3) << 3); };
  char* ws = (char*)ws_all[rank];
  lds_int* sync = (lds_int*)(smem + SYNC_OFF);
  unsigned* tmo = (unsigned*)(ws + WS_TMO);
  if (threadIdx.x < 160) sync[threadIdx.x] = 0;

  auto gk = [&](int kq) -> int {
    if (kq < 4) return rank * 16 + s + 4 * kq;
    int p = kq - 4;
    int j = (p * 9363) >> 16;
    int q = p - j * 7;
    return ((rank + 1 + q) & 7) * 16 + s + 4 * j;
  };

  if (wave < 8) {
    const int n0 = nt * 64;
    const unsigned aoff = (unsigned)(wave * 1024 + lane * 16);
    unsigned poff = 0;
    f32x4 ra[8], rb[8];
#pragma unroll
    for (int j = 0; j < 8; j++) { ra[j] = f32x4{0, 0, 0, 0}; rb[j] = f32x4{0, 0, 0, 0}; }
    asm volatile("s_waitcnt lgkmcnt(0)" ::: "memory");
    __builtin_amdgcn_s_barrier();
    unsigned call = 0;
    auto fetch = [&](int cc, int (&e)[8]) {
      const int first = cc < 12 ? (8 * cc) / 3 : 8 * (cc - 12);
      const int cnt = cc < 12 ? (8 * cc + 7) / 3 + 1 : 8 * (cc - 12) + 8;
      const int qi = cc < 12 ? 0 : 1;
      while (true) {
        int v = LDS_LD(lane == 0 ? qi : 4);
        bool ok = lane != 0 || v >= cnt;
        bool ab = lane == 1 && v != 0;
        if (__any(ab) || __all(ok)) break;
        __builtin_amdgcn_s_sleep(1);
      }
      asm volatile("" ::: "memory");
      const int q = LDS_LD(96 + qi * 32 + ((first + lane) & 31));
#pragma unroll
      for (int k = 0; k < 8; k++)
        if (k < cnt - first) e[k] = __builtin_amdgcn_readlane(q, k);
    };
    f32x4 acc[3][4];
#pragma unroll
    for (int q = 0; q < 3; q++)
#pragma unroll
      for (int n = 0; n < 4; n++) acc[q][n] = f32x4{0, 0, 0, 0};
    while ((call = (unsigned)__builtin_amdgcn_readfirstlane(LDS_LD(80))) == 0u) __builtin_amdgcn_s_sleep(1);
    const char* abase = ws + WS_A + (size_t)(call & 1) * A_BYTES;
#define ENT(cc, k) ((cc) < 12 ? (8 * (cc) + (k)) / 3 - (8 * (cc)) / 3 : (k))
#define REL(cc, k) ((cc) < 12 ? 1 + (8 * (cc) + (k)) % 3 : 0)
    auto aptr = [&](int ent, int rel) { return abase + (size_t)(ent & 127) * 32768 + (size_t)((s + rel) & 3) * 8192; };
    int e0[8], e1[8], e2[8];
    TS(1);
    fetch(0, e0);
#pragma unroll
    for (int k = 0; k < 8; k++) { const char* p = aptr(e0[ENT(0, k)], REL(0, k)); LDAR(ra[k], p); }
    fetch(1, e1);
#pragma unroll
    for (int k = 0; k < 8; k++) { const char* p = aptr(e1[ENT(1, k)], REL(1, k)); LDAR(rb[k], p); }
    TS(2);
    half8 b[4];
#define CHUNK(cc, R, CNT, EC, EN)                                                                             \
    {                                                                                                         \
      if ((cc) < 14) fetch((cc) + 2, EN);                                                                     \
      TS(8 + 2 * (cc));                                                                                       \
      _Pragma("unroll") for (int k = 0; k < 8; k++) {                                                         \
        if ((cc) >= 12 || (8 * (cc) + k) % 3 == 0) {                                                          \
          const lds_h8* sp = (const lds_h8*)(smem + (EC[ENT(cc, k)] >> 8) * 4096 + lane * 16);                \
          _Pragma("unroll") for (int n = 0; n < 4; n++) b[n] = sp[n * 64];                                    \
        }                                                                                                     \
        if ((CNT) >= 0) asm volatile("s_waitcnt vmcnt(%1)" : "+v"(R[k]) : "i"((CNT) >= 0 ? (CNT) : 0));       \
        _Pragma("unroll") for (int n = 0; n < 4; n++)                                                         \
            acc[(cc) < 12 ? (8 * (cc) + k) % 3 : 0][n] = __builtin_amdgcn_mfma_f32_16x16x32_f16(              \
                H8(R[k]), b[n], acc[(cc) < 12 ? (8 * (cc) + k) % 3 : 0][n], 0, 0, 0);                         \
        __builtin_amdgcn_sched_barrier(0);                                                                    \
        if ((cc) < 14) { const char* p = aptr(EN[ENT((cc) + 2, k)], REL((cc) + 2, k)); LDAR(R[k], p); }       \
        if ((cc) == 15 && k == 3) { _Pragma("unroll") for (int n = 0; n < 4; n++) LDPR(R[n], pin2, n * 1024); } \
      }                                                                                                       \
      TS(9 + 2 * (cc));                                                                                       \
    }
    const char* pin = ws + WS_PART + (size_t)bid * 3 * 32768;
    const char* pin1 = pin + 32768;
    const char* pin2 = pin + 65536;
    CHUNK(0, ra, 15, e0, e2); CHUNK(1, rb, 15, e1, e0); CHUNK(2, ra, 15, e2, e1);
    CHUNK(3, rb, 15, e0, e2); CHUNK(4, ra, 15, e1, e0); CHUNK(5, rb, 15, e2, e1);
    CHUNK(6, ra, 15, e0, e2); CHUNK(7, rb, 15, e1, e0); CHUNK(8, ra, 15, e2, e1);
    CHUNK(9, rb, 15, e0, e2); CHUNK(10, ra, 15, e1, e0); CHUNK(11, rb, 15, e2, e1);
    TS(3);
    int lz = __builtin_amdgcn_mbcnt_hi(~0u, __builtin_amdgcn_mbcnt_lo(~0u, 0u));
    asm volatile("" : "+v"(lz));
    poff = (unsigned)(wave * 4096 + lz * 16);
    int mode;
    while ((mode = __builtin_amdgcn_readfirstlane(LDS_LD(67))) == 0 && __builtin_amdgcn_readfirstlane(LDS_LD(4)) == 0)
      __builtin_amdgcn_s_sleep(1);
#pragma unroll
    for (int q = 0; q < 3; q++) {
      const char* p = ws + WS_PART + ((size_t)partner(q + 1) * 3 + q) * 32768;
      if (mode == 1) {
#pragma unroll
        for (int n = 0; n < 4; n++)
          asm volatile("global_store_dwordx4 %0, %1, %2 offset:%3\n\ts_nop 1" ::"v"(poff), "v"(acc[q][n]), "s"(p), "i"(n * 1024) : "memory");
      } else {
#pragma unroll
        for (int n = 0; n < 4; n++)
          asm volatile("global_store_dwordx4 %0, %1, %2 offset:%3 sc1\n\ts_nop 1" ::"v"(poff), "v"(acc[q][n]), "s"(p), "i"(n * 1024) : "memory");
      }
    }
#pragma unroll
    for (int n = 0; n < 4; n++) acc[0][n] = f32x4{0, 0, 0, 0};
    CHUNK(12, ra, 27, e0, e2);
    asm volatile("s_waitcnt vmcnt(8)" ::: "memory");
    {
      unsigned* pf = (unsigned*)(ws + WS_PFLAG + (size_t)partner((lz & 3) + 1) * 128 + (lz & 3) * 32 + wave * 4);
      if (lz < 3) asm volatile("global_store_dword %0, %1, off sc1" ::"v"(pf), "v"(call) : "memory");
    }
    TS(4);
    CHUNK(13, rb, -1, e1, e0);
    asm volatile("s_waitcnt vmcnt(9)" : REGS8(ra));
    CHUNK(14, ra, -1, e2, e1);
    bool bad = false;
    {
      const unsigned* pf = (const unsigned*)(ws + WS_PFLAG + (size_t)bid * 128 + (lz % 3) * 32 + wave * 4);
      for (unsigned it = 1; !__all((int)(__hip_atomic_load(pf, __ATOMIC_RELAXED, __HIP_MEMORY_SCOPE_AGENT) - call) >= 0); it++) {
        if ((it & 63) == 0 && (__hip_atomic_load(tmo, __ATOMIC_RELAXED, __HIP_MEMORY_SCOPE_AGENT) == call ||
                               __builtin_amdgcn_s_memrealtime() - t_start > DEADLINE_TICKS)) {
          __hip_atomic_store(tmo, call, __ATOMIC_RELAXED, __HIP_MEMORY_SCOPE_AGENT);
          bad = true;
          break;
        }
      }
    }
    asm volatile("s_waitcnt vmcnt(0)" : REGS8(rb));
    TS(5);
#pragma unroll
    for (int n = 0; n < 4; n++) LDPR(ra[n], pin, n * 1024);
#pragma unroll
    for (int n = 0; n < 4; n++) LDPR(ra[4 + n], pin1, n * 1024);
    CHUNK(15, rb, -1, e0, e1);
    asm volatile("s_waitcnt vmcnt(0)" : REGS8(ra));
    asm volatile("" : REGS8(rb));
    TS(6);
    bad = bad || LDS_LD(4) != 0;
#pragma unroll
    for (int n = 0; n < 4; n++) acc[0][n] = (acc[0][n] + ra[n]) + (ra[4 + n] + rb[n]);
#pragma unroll
    for (int r = 0; r < 4; r++) {
      f32x4 o;
#pragma unroll
      for (int n = 0; n < 4; n++) o[n] = bad ? __builtin_nanf("") : gelu_tanh(acc[0][n][r]);
      *(f32x4*)(C + (size_t)((s * 8 + wave) * 16 + (lz >> 4) * 4 + r) * ND + n0 + (lz & 15) * 4) = o;
    }
  } else if (wave > 9) {
    const int ld = wave - 10;
    unsigned woff[8];
#pragma unroll
    for (int j = 0; j < 8; j++) woff[j] = (unsigned)((g * 4 + (j >> 2) * 16 + (j & 3)) * ND + nt * 64 + c * 4) * 4u;
    f32x4 wr[WDEPTH][8];
#pragma unroll
    for (int q = 0; q < WDEPTH; q++)
#pragma unroll
      for (int j = 0; j < 8; j++) wr[q][j] = f32x4{0, 0, 0, 0};
    asm volatile("s_waitcnt lgkmcnt(0)" ::: "memory");
    __builtin_amdgcn_s_barrier();
    auto issue = [&](f32x4 (&w)[8], int kq) {
      const char* p = (const char*)W + (size_t)gk(kq) * (32 * ND * 4);
#pragma unroll
      for (int j = 0; j < 8; j++) LDWR(w[j], j, p);
    };
    constexpr int NK = 32 / NLOAD;
#pragma unroll
    for (int i = 0; i < WDEPTH; i++) {
      while (__builtin_amdgcn_s_memrealtime() - t_start < (unsigned long long)(WSTART + i * WRAMP)) __builtin_amdgcn_s_sleep(1);
      issue(wr[i], ld + i * NLOAD);
    }
#pragma unroll
    for (int i = 0; i < NK; i++) {
      const int kq = ld + i * NLOAD;
      const int left = NK - 1 - i < WDEPTH - 1 ? NK - 1 - i : WDEPTH - 1;
      int hv = LDS_LD(81);
      asm volatile("s_waitcnt vmcnt(%8)" : REGS8(wr[i % WDEPTH]) : "i"(left * 8));
      half8 b[4];
#pragma unroll
      for (int n = 0; n < 4; n++)
#pragma unroll
        for (int j = 0; j < 8; j++) b[n][j] = (_Float16)wr[i % WDEPTH][j][n];
      lds_h8* d = (lds_h8*)(smem + kq * 4096 + lane * 16);
#pragma unroll
      for (int n = 0; n < 4; n++) d[n * 64] = b[n];
      asm volatile("" ::: "memory");
      if (lane == 0) LDS_ST(2 + ld, i + 1);
      if (i + WDEPTH < NK) {
        while (hv != 0) { __builtin_amdgcn_s_sleep(1); hv = LDS_LD(81); }
        issue(wr[i % WDEPTH], kq + WDEPTH * NLOAD);
      }
    }
    TS(1);
  } else if (wave == 8) {
    const int d = bid & 7, n = bid >> 3;
    int tt[2], bb[2], ix[2];
#pragma unroll
    for (int e = 0; e < 2; e++) {
      const int u = 2 * n + e;
      int t, rel;
      if (u < 48) { t = (u * 21846) >> 16; rel = 1 + u - 3 * t; ix[e] = 3 * (t >> 2) + rel - 1; }
      else { t = u - 48; rel = 0; ix[e] = 12 + (t >> 2); }
      tt[e] = t; bb[e] = (t + rel) & 3;
    }
    f32x4 xv[32];
#pragma unroll
    for (int e = 0; e < 2; e++) {
      const float* xp = X + (size_t)(d * MS + bb[e] * 128 + c) * MS + tt[e] * 32 + g * 4;
#pragma unroll
      for (int r = 0; r < 8; r++) {
        xv[e * 16 + r * 2] = *(const f32x4*)(xp + (size_t)r * 16 * MS);
        xv[e * 16 + r * 2 + 1] = *(const f32x4*)(xp + (size_t)r * 16 * MS + 16);
      }
    }
    __builtin_amdgcn_sched_barrier(0);
    asm volatile("s_waitcnt lgkmcnt(0)" ::: "memory");
    __builtin_amdgcn_s_barrier();
    __builtin_amdgcn_sched_barrier(0);
    half8 h[16];
#pragma unroll
    for (int f = 0; f < 16; f++)
#pragma unroll
      for (int j = 0; j < 8; j++) h[f][j] = (_Float16)xv[f * 2 + (j >> 2)][j & 3];
    unsigned call;
    while ((call = (unsigned)__builtin_amdgcn_readfirstlane(LDS_LD(80))) == 0u) __builtin_amdgcn_s_sleep(1);
    TS(1);
    char* dws = (char*)ws_all[d];
    char* dbase = dws + WS_A + (size_t)(call & 1) * A_BYTES + (size_t)rank * 16 * 32768 + lane * 16;
    char* dst0 = dbase + (size_t)tt[0] * 32768 + bb[0] * 8192;
    char* dst1 = dbase + (size_t)tt[1] * 32768 + bb[1] * 8192;
    const int fe = (lane >> 1) & 1;
    unsigned* fl = (unsigned*)(dws + WS_FLAG + (((fe ? tt[1] : tt[0]) & 3) * 2 + (lane & 1)) * 512 + (rank * 16 + (fe ? ix[1] : ix[0])) * 4);
    const unsigned long long slot = (d == rank) ? 0ull : (unsigned long long)(SLOT_T0 + n * SLOT_DT);
    while (__builtin_amdgcn_s_memrealtime() - t_start + HOLD_LEAD < slot) __builtin_amdgcn_s_sleep(1);
    if (lane == 0) LDS_ST(81, 1);
    while (__builtin_amdgcn_s_memrealtime() - t_start < slot) __builtin_amdgcn_s_sleep(1);
    TS(2);
#pragma unroll
    for (int f = 0; f < 8; f++)
      asm volatile("global_store_dwordx4 %0, %1, off sc0 sc1\n\ts_nop 1" ::"v"(dst0 + f * 1024), "v"(h[f]) : "memory");
#pragma unroll
    for (int f = 0; f < 8; f++)
      asm volatile("global_store_dwordx4 %0, %1, off sc0 sc1\n\ts_nop 1" ::"v"(dst1 + f * 1024), "v"(h[8 + f]) : "memory");
    if (lane == 0) LDS_ST(81, 0);
    asm volatile("s_waitcnt vmcnt(0)" ::: "memory");
    if (lane < 4) __hip_atomic_store(fl, call, __ATOMIC_RELAXED, __HIP_MEMORY_SCOPE_SYSTEM);
    TS(3);
  } else {
    unsigned* cntp = (unsigned*)(ws + WS_CNT + (size_t)bid * 128);
    const unsigned call_v = __hip_atomic_load(cntp, __ATOMIC_RELAXED, __HIP_MEMORY_SCOPE_AGENT) + 1u;
    asm volatile("s_waitcnt lgkmcnt(0)" ::: "memory");
    __builtin_amdgcn_s_barrier();
    const unsigned call = __builtin_amdgcn_readfirstlane(call_v);
    if (lane == 0) LDS_ST(80, call);
    unsigned xcc;
    asm volatile("s_getreg_b32 %0, hwreg(HW_REG_XCC_ID, 0, 4)" : "=s"(xcc));
    const unsigned xtag = (call << 8) | xcc;
    if (lane == 0) __hip_atomic_store((unsigned*)(ws + WS_XCC + (size_t)bid * 128), xtag, __ATOMIC_RELAXED, __HIP_MEMORY_SCOPE_AGENT);
    const unsigned* myfl = (const unsigned*)(ws + WS_FLAG + (s * 2 + copy) * 512) + lane;
    const unsigned* xp = (const unsigned*)(ws + WS_XCC + (size_t)partner(lane & 3) * 128);
    const int my_gk = gk(lane & 31), my_src = my_gk >> 4, my_j = (my_gk & 15) >> 2;
    const unsigned my_bits = lane < 32 ? 7u << (3 * my_j) : 1u << (12 + my_j);
    const int my_code = my_gk | ((lane & 31) << 8);
    unsigned long long done = 0;
    int n1 = 0, n2 = 0;
    bool moded = false, fin = false;
#define PDEPTH 1
    constexpr int PD = PDEPTH;
    unsigned pv0[PD], pv1[PD], pvx[PD];
#define POLL_ISSUE(q)                                                                                          \
    asm volatile("global_load_dword %0, %3, off sc0 sc1\n\tglobal_load_dword %1, %3, off offset:256 sc0 sc1\n\t" \
                 "global_load_dword %2, %4, off sc1" : "+v"(pv0[q]), "+v"(pv1[q]), "+v"(pvx[q]) : "v"(myfl), "v"(xp))
#pragma unroll
    for (int q = 0; q < PD; q++) { pv0[q] = 0; pv1[q] = 0; pvx[q] = 0; }
#pragma unroll
    for (int q = 0; q < PD; q++) { POLL_ISSUE(q); __builtin_amdgcn_s_sleep(8); }
#pragma unroll 1
    for (unsigned it = 1; !fin; it++) {
#pragma unroll
      for (int q = 0; q < PD; q++) {
        asm volatile("s_waitcnt vmcnt(%3)" : "+v"(pv0[q]), "+v"(pv1[q]), "+v"(pvx[q]) : "i"(3 * (PD - 1)));
        unsigned long long ok0 = __ballot((int)(pv0[q] - call) >= 0), ok1 = __ballot((int)(pv1[q] - call) >= 0);
        const bool xall = __all((pvx[q] >> 8) == (call & 0xffffffu)), xsame = __all(pvx[q] == xtag);
        __builtin_amdgcn_sched_barrier(0);
        POLL_ISSUE(q);
        if (!moded && xall) {
          if (lane == 0) LDS_ST(67, xsame ? 1 : 2);
          moded = true;
        }
        const unsigned fb = (unsigned)(((my_src & 4) ? ok1 : ok0) >> (16 * (my_src & 3))) & 0xffffu;
        const int wdone = LDS_LD(2 + (lane & 1));
        const bool rdy = (fb & my_bits) == my_bits && wdone > ((lane & 31) >> 1);
        const unsigned long long nm = __ballot(rdy) & ~done;
        if (nm) {
          const unsigned half = lane < 32 ? (unsigned)nm : (unsigned)(nm >> 32);
          const int base = lane < 32 ? n1 : n2;
          if ((nm >> lane) & 1) LDS_ST(96 + (lane & 32) + base + __builtin_popcount(half & ((1u << (lane & 31)) - 1u)), my_code);
          n1 += __builtin_popcount((unsigned)nm);
          n2 += __builtin_popcount((unsigned)(nm >> 32));
          done |= nm;
          asm volatile("" ::: "memory");
          if (lane < 2) LDS_ST(lane, lane == 0 ? n1 : n2);
        }
      }
      if (moded && done == ~0ull) fin = true;
      if ((it & 63) == 0 && __builtin_amdgcn_s_memrealtime() - t_start > DEADLINE_TICKS / 2) {
        asm volatile("s_waitcnt vmcnt(0)" ::: "memory");
        if (__hip_atomic_load(tmo, __ATOMIC_RELAXED, __HIP_MEMORY_SCOPE_AGENT) == call ||
            __builtin_amdgcn_s_memrealtime() - t_start > DEADLINE_TICKS) {
          __hip_atomic_store(tmo, call, __ATOMIC_RELAXED, __HIP_MEMORY_SCOPE_AGENT);
          LDS_ST(4, 1);
          fin = true;
        }
      }
    }
    asm volatile("s_waitcnt vmcnt(0)" ::: "memory");
#pragma unroll
    for (int q = 0; q < PD; q++) asm volatile("" : "+v"(pv0[q]), "+v"(pv1[q]), "+v"(pvx[q]));
    TS(6);
    if (lane == 0) __hip_atomic_store(cntp, call, __ATOMIC_RELAXED, __HIP_MEMORY_SCOPE_AGENT);
  }
  TS_OUT;
}

extern "C" void kernel_launch(int rank, int nranks, ncclComm_t comm, hipStream_t stream, void* const* d_in,
                              const int* in_sizes, int n_in, void* d_out, int out_size, const int* in_dtypes,
                              int out_dtype, void* const* d_out_all, void* const* d_ws_all, size_t ws_bytes) {
  (void)comm; (void)in_sizes; (void)n_in; (void)out_size; (void)in_dtypes; (void)out_dtype; (void)d_out_all;
  if (nranks != NR || (unsigned)rank >= (unsigned)NR || ws_bytes < WS_NEED) {
    fprintf(stderr, "kernel_launch: unsupported configuration\n");
    return;
  }
  static bool init = false;
  if (!init) {
    hipError_t e = hipFuncSetAttribute((const void*)a2a_gemm_gelu_kernel, hipFuncAttributeMaxDynamicSharedMemorySize, LDS_BYTES);
    if (e != hipSuccess) { fprintf(stderr, "hipFuncSetAttribute: %s\n", hipGetErrorString(e)); return; }
    init = true;
  }
  a2a_gemm_gelu_kernel<<<256, 768, LDS_BYTES, stream>>>((const float*)d_in[0], (const float*)d_in[1], (float*)d_out,
                                                        d_ws_all, rank);
}
```

```cpp
#include <hip/hip_runtime.h>
#include <rccl/rccl.h>
#include <cstdio>

typedef _Float16 half8 __attribute__((ext_vector_type(8)));
typedef float f32x4 __attribute__((ext_vector_type(4)));
typedef __attribute__((address_space(3))) int lds_int;
typedef __attribute__((address_space(3))) half8 lds_h8;

namespace {
constexpr int NR = 8;
constexpr int ND = 4096;
constexpr int MS = 512;
constexpr size_t WS_TMO   = 0;
constexpr size_t WS_CNT   = 4096;
constexpr size_t WS_FLAG  = 64 * 1024;
constexpr size_t WS_PFLAG = 128 * 1024;
constexpr size_t WS_XCC   = 192 * 1024;
constexpr size_t WS_A     = 1 << 20;
constexpr size_t A_BYTES  = (size_t)4 << 20;
constexpr size_t WS_NEED  = WS_A + 2 * A_BYTES;
constexpr size_t WS_TS    = (size_t)40 << 20;
constexpr int SYNC_OFF = 32 * 4096;
constexpr int LDS_BYTES = SYNC_OFF + 1024;
constexpr unsigned long long DEADLINE_TICKS = 100000000ull;
constexpr int NLOAD = 2;
#define WDEPTH 3
#define HOLD_LEAD 50
#define SLOT_T0 150
#define SLOT_DT 28

__device__ __forceinline__ float gelu_tanh(float y) {
  float u = 0.7978845608028654f * (y + 0.044715f * y * y * y);
  float e = __expf(2.f * u);
  float t = 1.f - 2.f / (e + 1.f);
  return 0.5f * y * (1.f + t);
}
}

#define TS(k)
#define TS_DECL
#define TS_OUT

#define LDWR(dst, j, ptr) asm volatile("global_load_dwordx4 %0, %1, %2" : "+v"(dst) : "v"(woff[j]), "s"(ptr))
#define LDAR(dst, ptr) asm volatile("global_load_dwordx4 %0, %1, %2 sc0 sc1" : "+v"(dst) : "v"(aoff), "s"(ptr))
#define REGS8(r) "+v"(r[0]), "+v"(r[1]), "+v"(r[2]), "+v"(r[3]), "+v"(r[4]), "+v"(r[5]), "+v"(r[6]), "+v"(r[7])
#define H8(x) __builtin_bit_cast(half8, x)
#define LDS_LD(i) __hip_atomic_load(&sync[i], __ATOMIC_RELAXED, __HIP_MEMORY_SCOPE_WORKGROUP)
#define LDS_ST(i, v) __hip_atomic_store(&sync[i], (int)(v), __ATOMIC_RELAXED, __HIP_MEMORY_SCOPE_WORKGROUP)

__global__ __launch_bounds__(768, 3) void a2a_gemm_gelu_kernel(const float* __restrict__ X, const float* __restrict__ W,
                                                               float* __restrict__ C, void* const* __restrict__ ws_all,
                                                               int rank) {
  extern __shared__ __attribute__((aligned(16))) char smem[];
  TS_DECL; TS(0);
  const unsigned long long t_start = __builtin_amdgcn_s_memrealtime();
  const int wave = __builtin_amdgcn_readfirstlane(threadIdx.x >> 6), lane = threadIdx.x & 63;
  const int g = lane >> 4, c = lane & 15;
  const int bid = blockIdx.x;
#define XMAP 1
  const int s = bid & 1, mh = (bid >> 3) & 1, copy = (bid >> 1) & 1, nt = ((bid >> 1) & 3) + 4 * (bid >> 4);
  const int pbid = bid ^ 1;
  char* ws = (char*)ws_all[rank];
  lds_int* sync = (lds_int*)(smem + SYNC_OFF);
  unsigned* tmo = (unsigned*)(ws + WS_TMO);
  if (threadIdx.x < 96) sync[threadIdx.x] = 0;

  auto gk = [&](int kq) -> int {
    if (kq < 8) return rank * 16 + s + 2 * kq;
    int p = kq - 8;
    int j = (p * 9363) >> 16;
    int q = p - j * 7;
    return ((rank + 1 + q) & 7) * 16 + s + 2 * j;
  };

  if (wave < 8) {
    const int n0 = nt * 64;
    const unsigned aoff = (unsigned)(wave * 1024 + lane * 16);
    f32x4 ra[8], rb[8];
#pragma unroll
    for (int j = 0; j < 8; j++) { ra[j] = f32x4{0, 0, 0, 0}; rb[j] = f32x4{0, 0, 0, 0}; }
    asm volatile("s_waitcnt lgkmcnt(0)" ::: "memory");
    __builtin_amdgcn_s_barrier();
    unsigned call = 0;
    auto waitsync = [&](int pneed, int ownneed, int peerneed) {
      const int need = lane == 0 ? ownneed : (lane == 1 ? peerneed + 1 : (lane < 4 ? (pneed + NLOAD + 1 - lane) / NLOAD : 0));
      while (true) {
        int v = LDS_LD(lane & 7);
        bool ok = lane > 3 || v >= need;
        bool ab = lane == 4 && v != 0;
        if (__any(ab) || __all(ok)) break;
        __builtin_amdgcn_s_sleep(1);
      }
      asm volatile("" ::: "memory");
    };
    constexpr int OWNNEED[2] = {4, 8};
    constexpr int PEERNEED[16] = {-1, -1, 0, 1, 1, 2, 2, 3, 3, 4, 5, 5, 6, 6, 7, 7};
    f32x4 accS[4], accK[4];
#pragma unroll
    for (int n = 0; n < 4; n++) { accS[n] = f32x4{0, 0, 0, 0}; accK[n] = f32x4{0, 0, 0, 0}; }
    while ((call = (unsigned)__builtin_amdgcn_readfirstlane(LDS_LD(80))) == 0u) __builtin_amdgcn_s_sleep(1);
    const char* abase = ws + WS_A + (size_t)(call & 1) * A_BYTES + (size_t)mh * 16384;
    auto item_ptr = [&](int i) { return abase + (size_t)gk(i >> 1) * 32768 + (size_t)((i ^ s ^ 1) & 1) * 8192; };
    TS(1);
    waitsync(0, OWNNEED[0], PEERNEED[0]);
#pragma unroll
    for (int k = 0; k < 8; k++) { const char* p = item_ptr(k); LDAR(ra[k], p); }
    waitsync(0, OWNNEED[1], PEERNEED[1]);
#pragma unroll
    for (int k = 0; k < 8; k++) { const char* p = item_ptr(8 + k); LDAR(rb[k], p); }
    TS(2);
#define RE(cc) ((cc) >= 2 && (cc) < 14)
#define CHUNK(cc, R)                                                                                          \
    {                                                                                                         \
      waitsync(4 * (cc) + 4, 0, RE(cc) ? PEERNEED[RE(cc) ? (cc) + 2 : 0] : -1); \
      TS(8 + 2 * (cc));                                                                                       \
      const lds_h8* sp = (const lds_h8*)(smem + ((cc) & 7) * 4 * 4096 + lane * 16);                           \
      half8 b[2][4];                                                                                          \
      _Pragma("unroll") for (int n = 0; n < 4; n++) b[0][n] = sp[n * 64];                                     \
      _Pragma("unroll") for (int kk = 0; kk < 4; kk++) {                                                      \
        if (kk < 3) { _Pragma("unroll") for (int n = 0; n < 4; n++) b[(kk + 1) & 1][n] = sp[(kk + 1) * 256 + n * 64]; } \
        asm volatile("s_waitcnt vmcnt(%1)" : "+v"(R[2 * kk]) : "i"(RE(cc) ? 15 : (((cc) & 1) == 0 ? 15 - 2 * kk : 7 - 2 * kk))); \
        _Pragma("unroll") for (int n = 0; n < 4; n++)                                                         \
            accS[n] = __builtin_amdgcn_mfma_f32_16x16x32_f16(H8(R[2 * kk]), b[kk & 1][n], accS[n], 0, 0, 0);  \
        __builtin_amdgcn_sched_barrier(0);                                                                    \
        if (RE(cc)) { const char* p = item_ptr(((cc) + 2) * 8 + 2 * kk); LDAR(R[2 * kk], p); }                \
        asm volatile("s_waitcnt vmcnt(%1)" : "+v"(R[2 * kk + 1]) : "i"(RE(cc) ? 15 : (((cc) & 1) == 0 ? 14 - 2 * kk : 6 - 2 * kk))); \
        _Pragma("unroll") for (int n = 0; n < 4; n++)                                                         \
            accK[n] = __builtin_amdgcn_mfma_f32_16x16x32_f16(H8(R[2 * kk + 1]), b[kk & 1][n], accK[n], 0, 0, 0); \
        __builtin_amdgcn_sched_barrier(0);                                                                    \
        if (RE(cc)) { const char* p = item_ptr(((cc) + 2) * 8 + 2 * kk + 1); LDAR(R[2 * kk + 1], p); }        \
      }                                                                                                       \
      if ((cc) < 8 && lane == 0) LDS_ST(72 + wave, (cc) + 1);                                                 \
      TS(9 + 2 * (cc));                                                                                       \
    }
    CHUNK(0, ra); CHUNK(1, rb);
    waitsync(0, 0, PEERNEED[2]);
#pragma unroll
    for (int k = 0; k < 8; k++) { const char* p = item_ptr(16 + k); LDAR(ra[k], p); }
    waitsync(0, 0, PEERNEED[3]);
#pragma unroll
    for (int k = 0; k < 8; k++) { const char* p = item_ptr(24 + k); LDAR(rb[k], p); }
    CHUNK(2, ra); CHUNK(3, rb); CHUNK(4, ra); CHUNK(5, rb); CHUNK(6, ra); CHUNK(7, rb);
    CHUNK(8, ra); CHUNK(9, rb); CHUNK(10, ra); CHUNK(11, rb); CHUNK(12, ra); CHUNK(13, rb); CHUNK(14, ra); CHUNK(15, rb);
    TS(3);
    int mode;
    while ((mode = __builtin_amdgcn_readfirstlane(LDS_LD(67))) == 0 && __builtin_amdgcn_readfirstlane(LDS_LD(4)) == 0)
      __builtin_amdgcn_s_sleep(1);
    float* crow = C + (size_t)(mh * 256 + wave * 16 + g * 4) * ND + n0 + c * 4;
    float* cS = crow + (size_t)((s ^ 1) * 128) * ND;
    float* cK = crow + (size_t)(s * 128) * ND;
#pragma unroll
    for (int r = 0; r < 4; r++) {
      f32x4 o = {accS[0][r], accS[1][r], accS[2][r], accS[3][r]};
      if (mode == 1) asm volatile("global_store_dwordx4 %0, %1, off\n\ts_nop 1" ::"v"(cS + (size_t)r * ND), "v"(o) : "memory");
      else asm volatile("global_store_dwordx4 %0, %1, off sc1\n\ts_nop 1" ::"v"(cS + (size_t)r * ND), "v"(o) : "memory");
    }
    asm volatile("s_waitcnt vmcnt(0)" ::: "memory");
    if (lane == 0) __hip_atomic_store((unsigned*)(ws + WS_PFLAG + (size_t)pbid * 128 + wave * 4), call, __ATOMIC_RELAXED, __HIP_MEMORY_SCOPE_AGENT);
    TS(4);
    bool bad = false;
    {
      const unsigned* pf = (const unsigned*)(ws + WS_PFLAG + (size_t)bid * 128 + wave * 4);
      for (unsigned it = 1; (int)(__hip_atomic_load(pf, __ATOMIC_RELAXED, __HIP_MEMORY_SCOPE_AGENT) - call) < 0; it++) {
        if ((it & 63) == 0 && (__hip_atomic_load(tmo, __ATOMIC_RELAXED, __HIP_MEMORY_SCOPE_AGENT) == call ||
                               __builtin_amdgcn_s_memrealtime() - t_start > DEADLINE_TICKS)) {
          __hip_atomic_store(tmo, call, __ATOMIC_RELAXED, __HIP_MEMORY_SCOPE_AGENT);
          bad = true;
          break;
        }
      }
    }
    TS(5);
#pragma unroll
    for (int r = 0; r < 4; r++) asm volatile("global_load_dwordx4 %0, %1, off sc1" : "+v"(ra[r]) : "v"(cK + (size_t)r * ND));
    asm volatile("s_waitcnt vmcnt(0)" : REGS8(ra));
    TS(6);
    bad = bad || LDS_LD(4) != 0;
#pragma unroll
    for (int r = 0; r < 4; r++) {
      f32x4 o;
#pragma unroll
      for (int n = 0; n < 4; n++) o[n] = bad ? __builtin_nanf("") : gelu_tanh(accK[n][r] + ra[r][n]);
      *(f32x4*)(cK + (size_t)r * ND) = o;
    }
  } else if (wave > 9) {
    const int ld = wave - 10;
    unsigned woff[8];
#pragma unroll
    for (int j = 0; j < 8; j++) woff[j] = (unsigned)((g * 4 + (j >> 2) * 16 + (j & 3)) * ND + nt * 64 + c * 4) * 4u;
    f32x4 wr[WDEPTH][8];
#pragma unroll
    for (int q = 0; q < WDEPTH; q++)
#pragma unroll
      for (int j = 0; j < 8; j++) wr[q][j] = f32x4{0, 0, 0, 0};
    asm volatile("s_waitcnt lgkmcnt(0)" ::: "memory");
    __builtin_amdgcn_s_barrier();
    auto issue = [&](f32x4 (&w)[8], int kq) {
      const char* p = (const char*)W + (size_t)gk(kq) * (32 * ND * 4);
#pragma unroll
      for (int j = 0; j < 8; j++) LDWR(w[j], j, p);
    };
    const int cidx = lane < 8 ? 72 + lane : (lane == 10 ? 4 : 81);
    constexpr int NK = 64 / NLOAD;
#pragma unroll
    for (int i = 0; i < WDEPTH; i++) issue(wr[i], ld + i * NLOAD);
#pragma unroll
    for (int i = 0; i < NK; i++) {
      const int kq = ld + i * NLOAD;
      const int left = NK - 1 - i < WDEPTH - 1 ? NK - 1 - i : WDEPTH - 1;
      int cv = LDS_LD(cidx);
      asm volatile("s_waitcnt vmcnt(%8)" : REGS8(wr[i % WDEPTH]) : "i"(left * 8));
      half8 b[4];
#pragma unroll
      for (int n = 0; n < 4; n++)
#pragma unroll
        for (int j = 0; j < 8; j++) b[n][j] = (_Float16)wr[i % WDEPTH][j][n];
      const int need = i >= 32 / NLOAD ? ((kq - 32) >> 2) + 1 : 0;
      while (!__all(lane < 8 ? cv >= need : (lane != 9 || cv == 0)) && !__any(lane == 10 && cv != 0)) {
        __builtin_amdgcn_s_sleep(1);
        cv = LDS_LD(cidx);
      }
      if (i + WDEPTH < NK) issue(wr[i % WDEPTH], kq + WDEPTH * NLOAD);
      lds_h8* d = (lds_h8*)(smem + (kq & 31) * 4096 + lane * 16);
#pragma unroll
      for (int n = 0; n < 4; n++) d[n * 64] = b[n];
      asm volatile("" ::: "memory");
      if (lane == 0) LDS_ST(2 + ld, i + 1);
    }
    TS(1);
  } else if (wave == 8) {
    const int d = bid & 7, n = bid >> 3;
    const int t = n >> 1, uh = n & 1;
    const float* xp = X + (size_t)(d * MS + uh * 256 + c) * MS + t * 32 + g * 4;
    f32x4 xv[32];
#pragma unroll
    for (int r = 0; r < 16; r++) {
      xv[r * 2] = *(const f32x4*)(xp + (size_t)r * 16 * MS);
      xv[r * 2 + 1] = *(const f32x4*)(xp + (size_t)r * 16 * MS + 16);
    }
    __builtin_amdgcn_sched_barrier(0);
    asm volatile("s_waitcnt lgkmcnt(0)" ::: "memory");
    __builtin_amdgcn_s_barrier();
    __builtin_amdgcn_sched_barrier(0);
    half8 h[16];
#pragma unroll
    for (int f = 0; f < 16; f++)
#pragma unroll
      for (int j = 0; j < 8; j++) h[f][j] = (_Float16)xv[f * 2 + (j >> 2)][j & 3];
    unsigned call;
    while ((call = (unsigned)__builtin_amdgcn_readfirstlane(LDS_LD(80))) == 0u) __builtin_amdgcn_s_sleep(1);
    TS(1);
    char* dws = (char*)ws_all[d];
    char* dst = dws + WS_A + (size_t)(call & 1) * A_BYTES + (size_t)(rank * 16 + t) * 32768 + uh * 16384 + lane * 16;
    unsigned* fl = (unsigned*)(dws + WS_FLAG + ((uh * 2 + (t & 1)) * 2 + (lane & 1)) * 256 + (rank * 8 + (t >> 1)) * 4);
    const unsigned long long slot = (d == rank) ? 0ull : (unsigned long long)(SLOT_T0 + n * SLOT_DT);
    {
    while (__builtin_amdgcn_s_memrealtime() - t_start + HOLD_LEAD < slot) __builtin_amdgcn_s_sleep(1);
    if (lane == 0) LDS_ST(81, 1);
    while (__builtin_amdgcn_s_memrealtime() - t_start < slot) __builtin_amdgcn_s_sleep(1);
    TS(2);
#pragma unroll
    for (int f = 0; f < 16; f++)
      asm volatile("global_store_dwordx4 %0, %1, off sc0 sc1\n\ts_nop 1" ::"v"(dst + f * 1024), "v"(h[f]) : "memory");
    if (lane == 0) LDS_ST(81, 0);
    asm volatile("s_waitcnt vmcnt(0)" ::: "memory");
    if (lane < 2) __hip_atomic_store(fl, call, __ATOMIC_RELAXED, __HIP_MEMORY_SCOPE_SYSTEM);
    TS(3);
    }
  } else {
    unsigned* cntp = (unsigned*)(ws + WS_CNT + (size_t)bid * 128);
    const unsigned call_v = __hip_atomic_load(cntp, __ATOMIC_RELAXED, __HIP_MEMORY_SCOPE_AGENT) + 1u;
    asm volatile("s_waitcnt lgkmcnt(0)" ::: "memory");
    __builtin_amdgcn_s_barrier();
    const unsigned call = __builtin_amdgcn_readfirstlane(call_v);
    if (lane == 0) LDS_ST(80, call);
    unsigned xcc;
    asm volatile("s_getreg_b32 %0, hwreg(HW_REG_XCC_ID, 0, 4)" : "=s"(xcc));
    const unsigned xtag = (call << 8) | xcc;
    if (lane == 0) __hip_atomic_store((unsigned*)(ws + WS_XCC + (size_t)bid * 128), xtag, __ATOMIC_RELAXED, __HIP_MEMORY_SCOPE_AGENT);
    const char* myfl = ws + WS_FLAG + ((mh * 2 + s) * 2 + copy) * 256;
    const unsigned* xp = (const unsigned*)(ws + WS_XCC + (size_t)pbid * 128);
    bool moded = false;
    for (unsigned it = 1;; it++) {
      unsigned long long okm;
      {
        unsigned v = __hip_atomic_load((const unsigned*)myfl + lane, __ATOMIC_RELAXED, __HIP_MEMORY_SCOPE_SYSTEM);
        okm = __ballot((int)(v - call) >= 0);
      }
      if (!moded) {
        unsigned pv = __builtin_amdgcn_readfirstlane(__hip_atomic_load(xp, __ATOMIC_RELAXED, __HIP_MEMORY_SCOPE_AGENT));
        if ((pv >> 8) == (call & 0xffffffu)) {
          if (lane == 0) LDS_ST(67, pv == xtag ? 1 : 2);
          moded = true;
        }
      }
      int own = 0, peer = 8;
#pragma unroll
      for (int p = 0; p < 8; p++) {
        const int cnt = __builtin_ctz(~(unsigned)((okm >> (8 * p)) & 0xff));
        if (p == rank) own = cnt; else peer = cnt < peer ? cnt : peer;
      }
      if (lane < 2) LDS_ST(lane, lane == 0 ? own : peer);
      if (moded && okm == ~0ull) break;
      if ((it & 63) == 0 && (__hip_atomic_load(tmo, __ATOMIC_RELAXED, __HIP_MEMORY_SCOPE_AGENT) == call ||
                             __builtin_amdgcn_s_memrealtime() - t_start > DEADLINE_TICKS)) {
        __hip_atomic_store(tmo, call, __ATOMIC_RELAXED, __HIP_MEMORY_SCOPE_AGENT);
        LDS_ST(4, 1);
        break;
      }
      __builtin_amdgcn_s_sleep(32);
    }
    TS(6);
    if (lane == 0) __hip_atomic_store(cntp, call, __ATOMIC_RELAXED, __HIP_MEMORY_SCOPE_AGENT);
  }
  TS_OUT;
}

extern "C" void kernel_launch(int rank, int nranks, ncclComm_t comm, hipStream_t stream, void* const* d_in,
                              const int* in_sizes, int n_in, void* d_out, int out_size, const int* in_dtypes,
                              int out_dtype, void* const* d_out_all, void* const* d_ws_all, size_t ws_bytes) {
  (void)comm; (void)in_sizes; (void)n_in; (void)out_size; (void)in_dtypes; (void)out_dtype; (void)d_out_all;
  if (nranks != NR || (unsigned)rank >= (unsigned)NR || ws_bytes < WS_NEED) {
    fprintf(stderr, "kernel_launch: unsupported configuration\n");
    return;
  }
  static bool init = false;
  if (!init) {
    hipError_t e = hipFuncSetAttribute((const void*)a2a_gemm_gelu_kernel, hipFuncAttributeMaxDynamicSharedMemorySize, LDS_BYTES);
    if (e != hipSuccess) { fprintf(stderr, "hipFuncSetAttribute: %s\n", hipGetErrorString(e)); return; }
    init = true;
  }
  a2a_gemm_gelu_kernel<<<256, 768, LDS_BYTES, stream>>>((const float*)d_in[0], (const float*)d_in[1], (float*)d_out,
                                                        d_ws_all, rank);
}
```
